```python
import numpy as np

kernel_name = "gpu_gemm_ar_m2048_k2048_n2048_f32_none"


ABSMAX_RTOL = 0.001

COMPUTE_DTYPE = "float32"
EPILOGUE = "none"
IN_QUANT = None
OUT_QUANT = None
IN_DTYPES = {"x": "float32", "w_mat": "float32"}
WORLD_SIZE = 8

def setup_inputs():

    x = np.random.randn(2048, 2048).astype(np.float32)
    w_mat = (np.random.randn(2048, 2048) / np.sqrt(2048)).astype(np.float32)
    return {"x": x, "w_mat": w_mat}

def _epilogue(y):
    return y

def reference(x, w_mat):
    y = (x @ w_mat).astype(np.float32)
    return _epilogue(y)
```

```cpp
#include <hip/hip_runtime.h>
#include <rccl/rccl.h>
#include <cstdio>
#include <cstdint>
#include <type_traits>

#define HIP_CHECK(expr) do { hipError_t _e = (expr); if (_e != hipSuccess) { \
    fprintf(stderr, "HIP error %s at %s:%d\n", hipGetErrorString(_e), __FILE__, __LINE__); \
    return; } } while (0)

#define GAS __attribute__((address_space(1)))
#define LAS __attribute__((address_space(3)))
typedef float f32x4 __attribute__((ext_vector_type(4)));
typedef _Float16 f16x8 __attribute__((ext_vector_type(8)));
typedef unsigned u32x4 __attribute__((ext_vector_type(4)));
typedef unsigned long long u64;

#define SPW 1
#define ST_AUX 17
#define DEPTH 1
#define OUT_AUX 0
#define LD_AUX 17

constexpr int MM = 2048, NN = 2048, KSH = 256, NR = 8;
constexpr int NT = 8;
constexpr int NWG = 256;
constexpr int NGEMM = 4, NLOAD = 4, NSEND = SPW;
constexpr int NWAVES = NGEMM + NLOAD + NSEND;
constexpr int NTHREADS = NWAVES * 64;
constexpr int WPD = NWG * NSEND / NR;
constexpr int NPIECE = 2 * 32 * NT;
constexpr int PPW = NPIECE / WPD;

constexpr size_t CTR_OFF = 0;
constexpr size_t TMO_OFF = 64 << 10;
constexpr size_t FLAG_OFF = 128 << 10;
constexpr size_t FLAG_PAR = (size_t)NR * NT * 64 * 8;
constexpr size_t DBG_OFF = 256 << 10;
constexpr size_t DATA_OFF = 1 << 20;
constexpr size_t DATA_MAT = (size_t)64 * 32 * 4096;
constexpr size_t DATA_PAR = 2 * DATA_MAT;
constexpr size_t WS_NEED = DATA_OFF + 2 * DATA_PAR;

constexpr int SLOT = 16384;
constexpr int CTRL = NR * SLOT;
constexpr int LDS_BYTES = CTRL + 256;
constexpr long long DEADLINE = 50000000;

struct Params {
    const float* x;
    const float* w;
    float* out;
    char* ws[NR];
    int rank;
    int mode;
};

__device__ __forceinline__ u64 rtc() { return __builtin_amdgcn_s_memrealtime(); }

__device__ __forceinline__ float wave_max(float v) {
    int x = __builtin_bit_cast(int, v);
    x = max(x, __builtin_amdgcn_update_dpp(0, x, 0xB1, 0xf, 0xf, true));
    x = max(x, __builtin_amdgcn_update_dpp(0, x, 0x4E, 0xf, 0xf, true));
    x = max(x, __builtin_amdgcn_update_dpp(0, x, 0x141, 0xf, 0xf, true));
    x = max(x, __builtin_amdgcn_update_dpp(0, x, 0x140, 0xf, 0xf, true));
    const int a = max(__builtin_amdgcn_readlane(x, 0), __builtin_amdgcn_readlane(x, 16));
    const int b = max(__builtin_amdgcn_readlane(x, 32), __builtin_amdgcn_readlane(x, 48));
    return __builtin_bit_cast(float, max(a, b));
}

#define STAMP(idx) do {} while (0)

__global__ void __launch_bounds__(NTHREADS, 1) fused_gemm_ar(Params p) {
    extern __shared__ __attribute__((aligned(16))) unsigned char lds[];
    LAS unsigned char* L8 = (LAS unsigned char*)lds;
    volatile LAS unsigned* FULL = (volatile LAS unsigned*)(L8 + CTRL);
    volatile LAS unsigned* FREE = FULL + 32;
    volatile LAS unsigned* ABORT = FULL + 48;

    const int tid = threadIdx.x, lane = tid & 63;
    const int wave = __builtin_amdgcn_readfirstlane(tid >> 6);
    const int wg = blockIdx.x, xcd = wg & 7, sl = wg >> 3;
    char* myws = p.ws[p.rank];

    if (wave >= NGEMM + NLOAD) {
        if (!(p.mode & 1)) { __builtin_amdgcn_s_barrier(); return; }
        STAMP(0);
        const int sw = wave - (NGEMM + NLOAD);
        const int dd = sl & 7;
        const int wi = (xcd * 4 + (sl >> 3)) * NSEND + sw;
        const unsigned dest = (unsigned)(p.rank + dd) & 7u;
        GAS unsigned* ctr = (GAS unsigned*)(myws + CTR_OFF) + wg * 16 + wave;
        const int r16 = lane & 15, g = lane >> 4;

        auto run = [&](auto mat_c) {
        constexpr int MAT = decltype(mat_c)::value;
        f32x4 b[8];
        f16x8 o[4];
        auto piece_of = [&](int i, int& mat, int& j, int& t) {
            const int q = wi + i * WPD;
            t = q >> 6; mat = MAT; j = (q & 63) >> 1;
        };
        auto load_piece = [&](int i) {
            int mat, j, t; piece_of(i, mat, j, t);
            if (MAT == 0) {
                const float* src = p.x + (size_t)(64 * j + r16) * KSH + 32 * t + 8 * g;
#pragma unroll
                for (int u = 0; u < 8; ++u) b[u] = *(const f32x4*)(src + (u >> 1) * 16 * KSH + (u & 1) * 4);
            } else {
                const float* src = p.w + (size_t)(32 * t + 8 * g) * NN + 64 * j + 4 * r16;
#pragma unroll
                for (int u = 0; u < 8; ++u) b[u] = *(const f32x4*)(src + (size_t)u * NN);
            }
        };
        auto convert = [&](int i) -> int {
            int mat, j, t; piece_of(i, mat, j, t);
            float am = 0.f;
#pragma unroll
            for (int u = 0; u < 8; ++u)
                am = fmaxf(fmaxf(fmaxf(fabsf(b[u][0]), fabsf(b[u][1])), fmaxf(fabsf(b[u][2]), fabsf(b[u][3]))), am);
            am = wave_max(am);
            const int ex = __builtin_amdgcn_readfirstlane(__builtin_amdgcn_frexp_expf(am));
            const int top = ((ex + 1) >> 3) * 8 + 6;
            const int e = 15 - top;
            if (MAT == 0) {
#pragma unroll
                for (int f = 0; f < 4; ++f)
#pragma unroll
                    for (int c = 0; c < 8; ++c) o[f][c] = (_Float16)__builtin_amdgcn_ldexpf(b[2 * f + (c >> 2)][c & 3], e);
            } else {
#pragma unroll
                for (int f = 0; f < 4; ++f)
#pragma unroll
                    for (int c = 0; c < 8; ++c) o[f][c] = (_Float16)__builtin_amdgcn_ldexpf(b[c][f], e);
            }
            return e;
        };
        load_piece(0);
        asm volatile("" ::: "memory");
        const unsigned call = __hip_atomic_load(ctr, __ATOMIC_RELAXED, __HIP_MEMORY_SCOPE_AGENT) + 1u;
        __builtin_amdgcn_s_barrier();
        int ecur = convert(0), eprev = 0;
        if (lane == 0) __hip_atomic_store(ctr, call, __ATOMIC_RELAXED, __HIP_MEMORY_SCOPE_AGENT);
        const unsigned par = call & 1u;
        char* dws = p.ws[dest];
        const __amdgpu_buffer_rsrc_t rs = __builtin_amdgcn_make_buffer_rsrc(dws + DATA_OFF + par * DATA_PAR, 0, (int)DATA_PAR, 0x00020000);
        GAS u64* dflag = (GAS u64*)(dws + FLAG_OFF + par * FLAG_PAR) + (size_t)p.rank * NT * 64;
        auto publish = [&](int i, int e) {
            int mat, j, t; piece_of(i, mat, j, t);
            if (lane == 0)
                __hip_atomic_store(dflag + t * 64 + mat * 32 + j, ((u64)call << 32) | (unsigned)e, __ATOMIC_RELAXED, __HIP_MEMORY_SCOPE_SYSTEM);
            STAMP(1 + i * 8 / PPW);
        };
#pragma unroll 1
        for (int i = 0; i < PPW; ++i) {
            int mat, j, t; piece_of(i, mat, j, t);
            asm volatile("" ::: "memory");
            if (i + 1 < PPW) load_piece(i + 1);
            asm volatile("" ::: "memory");
            const int off = (int)(mat * DATA_MAT) + ((p.rank * NT + t) * 32 + j) * 4096 + lane * 16;
#pragma unroll
            for (int f = 0; f < 4; ++f)
                __builtin_amdgcn_raw_buffer_store_b128(__builtin_bit_cast(u32x4, o[f]), rs, off + f * 1024, 0, ST_AUX);
            asm volatile("s_waitcnt vmcnt(4)" ::: "memory");
            eprev = ecur;
            if (i + 1 < PPW) ecur = convert(i + 1);
            asm volatile("s_waitcnt vmcnt(0)" ::: "memory");
            publish(i, eprev);
        }
        };
        if (wi & 1) run(std::integral_constant<int, 1>{}); else run(std::integral_constant<int, 0>{});
        return;
    }

    if (tid < 64) FULL[lane] = 0;
    asm volatile("s_waitcnt lgkmcnt(0)" ::: "memory");
    __builtin_amdgcn_s_barrier();
    if (!(p.mode & 2)) return;

    const int tr = (xcd >> 1) * 4 + (sl >> 3), tc = (xcd & 1) * 8 + (sl & 7);

    if (wave >= NGEMM) {
        const int lw = wave - NGEMM;
        GAS unsigned* ctr = (GAS unsigned*)(myws + CTR_OFF) + wg * 16 + wave;
        const unsigned call = __hip_atomic_load(ctr, __ATOMIC_RELAXED, __HIP_MEMORY_SCOPE_AGENT) + 1u;
        if (lane == 0) __hip_atomic_store(ctr, call, __ATOMIC_RELAXED, __HIP_MEMORY_SCOPE_AGENT);
        GAS unsigned* tmo = (GAS unsigned*)(myws + TMO_OFF);
        const unsigned par = call & 1u;
        const char* data = myws + DATA_OFF + par * DATA_PAR;
        GAS u64* flags = (GAS u64*)(myws + FLAG_OFF + par * FLAG_PAR);
        const int half = (lane >> 2) & 1, fi = lane & 3;
        const int myslot = 2 * lw + half;
        const int mysrc = (p.rank + myslot) & 7;
        const int fidx = (fi >> 1) * 32 + ((fi >> 1) ? 2 * tc : 2 * tr) + (fi & 1);
        int tt[2] = {0, 0};
        const u64 t0 = rtc();
        int spins = 0;
        while (tt[0] < NT || tt[1] < NT) {
            const int myt = half ? tt[1] : tt[0];
            const int tcl = myt < NT ? myt : NT - 1;
            const u64 f = __hip_atomic_load(flags + (size_t)(mysrc * NT + tcl) * 64 + fidx, __ATOMIC_RELAXED, __HIP_MEMORY_SCOPE_SYSTEM);
            const unsigned fr = FREE[myslot];
            const bool ok = ((unsigned)(f >> 32) == call) && (fr >= 4u * (unsigned)myt) && (myt < NT);
            const unsigned m = (unsigned)__ballot(ok);
            const bool ra = (m & 0xFu) == 0xFu, rb = (m & 0xF0u) == 0xF0u;
            if (ra || rb) {
                asm volatile("" ::: "memory");
#pragma unroll
                for (int h = 0; h < 2; ++h) {
                    if (h ? rb : ra) {
                        const int s = 2 * lw + h, src = (p.rank + s) & 7;
                        const size_t ks = (size_t)(src * NT + tt[h]) * 32;
                        const char* gx = data + (ks + 2 * tr) * 4096 + lane * 16;
                        const char* gw = data + DATA_MAT + (ks + 2 * tc) * 4096 + lane * 16;
#pragma unroll
                        for (int u = 0; u < 8; ++u)
                            __builtin_amdgcn_global_load_lds((const unsigned*)(gx + u * 1024), (unsigned*)(lds + s * SLOT + u * 1024), 16, 0, LD_AUX);
#pragma unroll
                        for (int u = 0; u < 8; ++u)
                            __builtin_amdgcn_global_load_lds((const unsigned*)(gw + u * 1024), (unsigned*)(lds + s * SLOT + 8192 + u * 1024), 16, 0, LD_AUX);
                    }
                }
                const int ev = (int)(unsigned)f;
                const int esum = __shfl(ev, half * 4 + (fi >> 1), 64) + __shfl(ev, half * 4 + 2 + (fi & 1), 64);
                asm volatile("s_waitcnt vmcnt(0)" ::: "memory");
                if (lane < 8 && (half ? rb : ra)) FULL[myslot * 4 + fi] = ((unsigned)(myt + 1) << 16) | ((unsigned)esum & 0xFFFFu);
                if (ra) { tt[0]++; STAMP(tt[0] - 1); }
                if (rb) { tt[1]++; STAMP(8 + tt[1] - 1); }
            } else {
                __builtin_amdgcn_s_sleep(2);
                if ((++spins & 63) == 0) {
                    if (__builtin_amdgcn_readfirstlane(*ABORT) != 0u) break;
                    if (__hip_atomic_load(tmo, __ATOMIC_RELAXED, __HIP_MEMORY_SCOPE_AGENT) == call || (long long)(rtc() - t0) > DEADLINE) {
                        if (lane == 0) { __hip_atomic_store(tmo, call, __ATOMIC_RELAXED, __HIP_MEMORY_SCOPE_AGENT); *ABORT = 1u; }
                        break;
                    }
                }
            }
        }
        return;
    }

    {
        STAMP(0);
        const int wr = wave >> 1, wc = wave & 1;
        f32x4 acc[4][4];
#pragma unroll
        for (int m = 0; m < 4; ++m)
#pragma unroll
            for (int c = 0; c < 4; ++c) acc[m][c] = (f32x4){0.f, 0.f, 0.f, 0.f};
        const __amdgpu_buffer_rsrc_t ro = __builtin_amdgcn_make_buffer_rsrc(p.out, 0, MM * NN * 4, 0x00020000);
        const int ooff = ((tr * 128 + wr * 64 + (lane >> 4) * 4) * NN + tc * 128 + wc * 64 + (lane & 15) * 4) * 4;
        unsigned done = 0;
        int remaining = NR * NT;
        bool flushed = false, aborted = false;
        int rr = 0;
        const unsigned abase = (unsigned)(wr * 4096 + lane * 16), bbase = (unsigned)(8192 + wc * 4096 + lane * 16);
        volatile LAS unsigned* myfull = FULL + (lane & 7) * 4 + wave;
        f16x8 fa[2][4], fb[2][4];
        auto pick = [&](unsigned fl, int& e) -> int {
            const unsigned m8 = (unsigned)__ballot((fl >> 16) > done) & 0xFFu;
            if (m8 == 0u) return -1;
            const unsigned rot = ((m8 >> rr) | (m8 << (8 - rr))) & 0xFFu;
            const int s = (rr + __builtin_ctz(rot)) & 7;
            e = (int)(short)(__builtin_amdgcn_readlane((int)fl, s) & 0xFFFF);
            return s;
        };
        auto fetch = [&](int s, f16x8* a, f16x8* b) {
            const LAS unsigned char* sb = L8 + s * SLOT;
#pragma unroll
            for (int m = 0; m < 4; ++m) a[m] = *(const LAS f16x8*)(sb + abase + m * 1024);
#pragma unroll
            for (int c = 0; c < 4; ++c) b[c] = *(const LAS f16x8*)(sb + bbase + c * 1024);
            rr = (s + 1) & 7;
            if ((lane & 7) == s) done++;
            remaining--;
        };
        auto release = [&](int s, f16x8* a, f16x8* b) {
            asm volatile("s_waitcnt lgkmcnt(0)" : "+v"(a[0]), "+v"(a[1]), "+v"(a[2]), "+v"(a[3]), "+v"(b[0]), "+v"(b[1]), "+v"(b[2]), "+v"(b[3]) :: "memory");
            if (lane == 0) __hip_atomic_fetch_add((LAS unsigned*)(FREE + s), 1u, __ATOMIC_RELAXED, __HIP_MEMORY_SCOPE_WORKGROUP);
        };
        auto mma = [&](int m0, const f16x8* a, const f16x8* b) {
#pragma unroll
            for (int m = m0; m < m0 + 2; ++m)
#pragma unroll
                for (int c = 0; c < 4; ++c)
                    acc[m][c] = __builtin_amdgcn_mfma_f32_16x16x32_f16(a[m], b[c], acc[m][c], 0, 0, 0);
        };
        while (remaining > 0 && !aborted) {
            int ecur = 0, s;
            for (;;) {
                s = pick(*myfull, ecur);
                if (s >= 0) break;
                if (__builtin_amdgcn_readfirstlane(*ABORT) != 0u) { aborted = true; break; }
                __builtin_amdgcn_s_sleep(1);
            }
            if (aborted) break;
            asm volatile("" ::: "memory");
            fetch(s, fa[0], fb[0]);
            release(s, fa[0], fb[0]);
            bool more = true;
            while (more) {
#pragma unroll
                for (int h = 0; h < 2; ++h) {
                    if (more) {
                        unsigned fl = *myfull;
                        mma(0, fa[h], fb[h]);
                        int e = ecur;
                        s = remaining > 0 ? pick(fl, e) : -1;
                        const bool got = s >= 0 && e == ecur;
                        asm volatile("" ::: "memory");
                        if (got) fetch(s, fa[h ^ 1], fb[h ^ 1]);
                        mma(2, fa[h], fb[h]);
                        if (got) release(s, fa[h ^ 1], fb[h ^ 1]);
                        else if (remaining == 0 || s >= 0) more = false;
                        else {
                            for (;;) {
                                s = pick(*myfull, e);
                                if (s >= 0) break;
                                if (__builtin_amdgcn_readfirstlane(*ABORT) != 0u) { aborted = true; break; }
                                __builtin_amdgcn_s_sleep(1);
                            }
                            if (aborted || e != ecur) more = false;
                            else { asm volatile("" ::: "memory"); fetch(s, fa[h ^ 1], fb[h ^ 1]); release(s, fa[h ^ 1], fb[h ^ 1]); }
                        }
                    }
                }
            }
            if (remaining == 0) { STAMP(1); }
#pragma unroll
            for (int m = 0; m < 4; ++m)
#pragma unroll
                for (int r = 0; r < 4; ++r) {
                    f32x4 v;
#pragma unroll
                    for (int c = 0; c < 4; ++c) { v[c] = __builtin_amdgcn_ldexpf(acc[m][c][r], -ecur); acc[m][c][r] = 0.f; }
                    if (flushed) v += __builtin_bit_cast(f32x4, __builtin_amdgcn_raw_buffer_load_b128(ro, ooff, (m * 16 + r) * NN * 4, 0));
                    __builtin_amdgcn_raw_buffer_store_b128(__builtin_bit_cast(u32x4, v), ro, ooff, (m * 16 + r) * NN * 4, OUT_AUX);
                }
            flushed = true;
        }
        if (aborted) {
            const f32x4 v = {__builtin_nanf(""), __builtin_nanf(""), __builtin_nanf(""), __builtin_nanf("")};
#pragma unroll
            for (int m = 0; m < 16; ++m)
                __builtin_amdgcn_raw_buffer_store_b128(__builtin_bit_cast(u32x4, v), ro, ooff, ((m >> 2) * 16 + (m & 3)) * NN * 4, 0);
        }
    }
}

extern "C" void kernel_launch(int rank, int nranks, ncclComm_t comm,
                              hipStream_t stream,
                              void* const* d_in, const int* in_sizes, int n_in,
                              void* d_out, int out_size,
                              const int* in_dtypes, int out_dtype,
                              void* const* d_out_all,
                              void* const* d_ws_all, size_t ws_bytes) {
    (void)comm; (void)in_dtypes; (void)out_dtype; (void)d_out_all;
    if (nranks != NR || n_in != 2 || (unsigned)rank >= (unsigned)NR || ws_bytes < WS_NEED ||
        in_sizes[0] != MM * KSH || in_sizes[1] != KSH * NN || out_size != MM * NN) {
        fprintf(stderr, "kernel_launch: unexpected problem geometry\n");
        return;
    }
    HIP_CHECK(hipFuncSetAttribute((const void*)fused_gemm_ar, hipFuncAttributeMaxDynamicSharedMemorySize, LDS_BYTES));
    Params p;
    p.x = (const float*)d_in[0];
    p.w = (const float*)d_in[1];
    p.out = (float*)d_out;
    for (int i = 0; i < NR; ++i) p.ws[i] = (char*)d_ws_all[i];
    p.rank = rank;
    p.mode = 3;
    hipLaunchKernelGGL(fused_gemm_ar, dim3(NWG), dim3(NTHREADS), LDS_BYTES, stream, p);
    HIP_CHECK(hipGetLastError());
}
```

```cpp
#include <hip/hip_runtime.h>
#include <rccl/rccl.h>
#include <cstdio>
#include <cstdint>
#include <type_traits>

#define HIP_CHECK(expr) do { hipError_t _e = (expr); if (_e != hipSuccess) { \
    fprintf(stderr, "HIP error %s at %s:%d\n", hipGetErrorString(_e), __FILE__, __LINE__); \
    return; } } while (0)

#define GAS __attribute__((address_space(1)))
#define LAS __attribute__((address_space(3)))
typedef float f32x4 __attribute__((ext_vector_type(4)));
typedef _Float16 f16x8 __attribute__((ext_vector_type(8)));
typedef unsigned u32x4 __attribute__((ext_vector_type(4)));
typedef unsigned u32x2 __attribute__((ext_vector_type(2)));
typedef unsigned long long u64;

#define SDIV 1
#define SFIRST 1
#define SPRIO 3
#define POLL_SLOW 48
#define POLL_THR 48
#define PD 1
#define POLL_AUX 17
#define SMAP 0
#define EARLY 0
#define ROT 4
#define ST_AUX 17
#define WIN 2
#define NSLOT 9
#define POLL_SLEEP 1
#define OUT_AUX 16
#define LD_AUX 17

constexpr int MM = 2048, NN = 2048, KSH = 256, NR = 8;
constexpr int NT = 8;
constexpr int NWG = 256;
constexpr int NGEMM = 4, NLOAD = 4;
constexpr int NWAVES = NGEMM + NLOAD + 1;
constexpr int NTHREADS = NWAVES * 64;
constexpr int WPD = NWG / SDIV / NR;
constexpr int WPM = WPD / 2;
constexpr int NPM = 32 * NT;

constexpr size_t CTR_OFF = 0;
constexpr size_t NXT_OFF = 32 << 10;
constexpr size_t TMO_OFF = 64 << 10;
constexpr size_t FLAG_OFF = 128 << 10;
constexpr size_t FLAG_PAR = (size_t)NR * NT * 64 * 2;
constexpr size_t DBG_OFF = 256 << 10;
constexpr size_t DATA_OFF = 1 << 20;
constexpr size_t DATA_MAT = (size_t)64 * 32 * 4096;
constexpr size_t DATA_PAR = 2 * DATA_MAT;
constexpr size_t LAST_OFF = 2 * DATA_PAR;
constexpr size_t LAST_BUF = (size_t)2 * NR * 32 * 4096;
constexpr size_t DATA_ALL = LAST_OFF + 3 * LAST_BUF;
constexpr size_t WS_NEED = DATA_OFF + DATA_ALL;

constexpr int SLOT = 16384;
constexpr int CTRL = NSLOT * SLOT;
constexpr int LDS_BYTES = CTRL + 5120;
constexpr long long DEADLINE = 50000000;

struct Params {
    const float* x;
    const float* w;
    float* out;
    char* ws[NR];
    int rank;
    int mode;
};

__device__ __forceinline__ u64 rtc() { return __builtin_amdgcn_s_memrealtime(); }

__device__ __forceinline__ float wave_max(float v) {
    int x = __builtin_bit_cast(int, v);
    x = max(x, __builtin_amdgcn_update_dpp(0, x, 0xB1, 0xf, 0xf, true));
    x = max(x, __builtin_amdgcn_update_dpp(0, x, 0x4E, 0xf, 0xf, true));
    x = max(x, __builtin_amdgcn_update_dpp(0, x, 0x141, 0xf, 0xf, true));
    x = max(x, __builtin_amdgcn_update_dpp(0, x, 0x140, 0xf, 0xf, true));
    const int a = max(__builtin_amdgcn_readlane(x, 0), __builtin_amdgcn_readlane(x, 16));
    const int b = max(__builtin_amdgcn_readlane(x, 32), __builtin_amdgcn_readlane(x, 48));
    return __builtin_bit_cast(float, max(a, b));
}

#define STAMP(idx) do {} while (0)
#define STAMP_DUMP() do {} while (0)

template <int N> __device__ __forceinline__ void wait_vm() { asm volatile("s_waitcnt vmcnt(%0)" :: "n"(N) : "memory"); }
constexpr int win_cnt(int f) { return (WIN - 1) + 9 * ((WIN - f + 3) / 4); }

__global__ void __launch_bounds__(NTHREADS, 1) fused_gemm_ar(Params p) {
    extern __shared__ __attribute__((aligned(16))) unsigned char lds[];
    LAS unsigned char* L8 = (LAS unsigned char*)lds;
    volatile LAS unsigned* RQ = (volatile LAS unsigned*)(L8 + CTRL);
    LAS u64* AVAIL = (LAS u64*)(L8 + CTRL + 1088);
    volatile LAS unsigned char* FIN = (volatile LAS unsigned char*)(L8 + CTRL + 1100);
    volatile LAS unsigned char* ABORT = FIN + 1;
    const unsigned cbase = (unsigned)(uintptr_t)(L8 + CTRL);

    const int tid = threadIdx.x, lane = tid & 63;
    const int hwave = __builtin_amdgcn_readfirstlane(tid >> 6);
    const int wave = SFIRST ? (hwave == 0 ? NWAVES - 1 : hwave - 1) : hwave;
    const int wg = blockIdx.x, xcd = wg & 7, sl = wg >> 3;
    char* myws = p.ws[p.rank];

    if (wave == NGEMM + NLOAD) {
        if (!(p.mode & 1) || (sl >> 3) % SDIV) { __builtin_amdgcn_s_barrier(); return; }
        __builtin_amdgcn_s_setprio(SPRIO);
        const int dd = SMAP ? xcd : sl & 7;
        const int wi = SMAP ? (sl & 7) * (4 / SDIV) + (sl >> 3) / SDIV
                            : xcd * (4 / SDIV) + (sl >> 3) / SDIV;
        const unsigned dest = (unsigned)(p.rank + dd) & 7u;
        GAS unsigned* ctr = (GAS unsigned*)(myws + CTR_OFF) + wg * 16 + wave;
        const int r16 = lane & 15, g = lane >> 4;

        auto run = [&](auto mat_c) {
            constexpr int MAT = decltype(mat_c)::value;
            f32x4 b[2][8];
            f16x8 o[4];
            auto piece_of = [&](int q, int& j, int& t) {
                q = q < NPM ? q : NPM - 1;
                t = q >> 5; j = (q + ROT * dd) & 31;
            };
            auto load_piece = [&](int q, f32x4* bb) {
                int j, t; piece_of(q, j, t);
                if (MAT == 0) {
                    const float* src = p.x + (size_t)(64 * j + r16) * KSH + 32 * t + 8 * g;
#pragma unroll
                    for (int u = 0; u < 8; ++u) bb[u] = *(const f32x4*)(src + (u >> 1) * 16 * KSH + (u & 1) * 4);
                } else {
                    const float* src = p.w + (size_t)(32 * t + 8 * g) * NN + 64 * j + 4 * r16;
#pragma unroll
                    for (int u = 0; u < 8; ++u) bb[u] = *(const f32x4*)(src + (size_t)u * NN);
                }
            };
            auto convert = [&](const f32x4* bb) -> int {
                float am = 0.f;
#pragma unroll
                for (int u = 0; u < 8; ++u)
                    am = fmaxf(fmaxf(fmaxf(fabsf(bb[u][0]), fabsf(bb[u][1])), fmaxf(fabsf(bb[u][2]), fabsf(bb[u][3]))), am);
                am = wave_max(am);
                const int ex = __builtin_amdgcn_frexp_expf(am);
                const int m8 = (ex + 1) >> 3;
                const int e = 9 - 8 * m8;
#pragma unroll
                for (int f = 0; f < 4; ++f)
#pragma unroll
                    for (int c = 0; c < 8; ++c)
                        o[f][c] = (_Float16)__builtin_amdgcn_ldexpf(MAT == 0 ? bb[2 * f + (c >> 2)][c & 3] : bb[c][f], e);
#pragma unroll
                for (int f = 0; f < (EARLY ? 4 : 0); ++f) {
                    u32x4 ov = __builtin_bit_cast(u32x4, o[f]);
                    ov[0] = ov[0] ? ov[0] : 0x8000u;
                    o[f] = __builtin_bit_cast(f16x8, ov);
                }
                return m8;
            };
            int q0 = wi >> 1, q1 = q0 + WPM, q2 = q1 + WPM, qp = 0;
            load_piece(q0, b[0]);
            asm volatile("" ::: "memory");
            const unsigned call = __hip_atomic_load(ctr, __ATOMIC_RELAXED, __HIP_MEMORY_SCOPE_AGENT) + 1u;
            asm volatile("" ::: "memory");
            load_piece(q1, b[1]);
            asm volatile("" ::: "memory");
            STAMP(0);
            int e0 = convert(b[0]), e1 = 0;
            const unsigned par = call & 1u;
            char* dws = p.ws[dest];
            const __amdgpu_buffer_rsrc_t rs = __builtin_amdgcn_make_buffer_rsrc(dws + DATA_OFF, 0, (int)DATA_ALL, 0x00020000);
            const int obase = (int)(par * DATA_PAR + MAT * DATA_MAT) + p.rank * NT * 32 * 4096;
            const int olast = (int)(LAST_OFF + (call % 3u) * LAST_BUF) + (MAT * NR + p.rank) * 32 * 4096;
            GAS unsigned short* dflag = (GAS unsigned short*)(dws + FLAG_OFF + par * FLAG_PAR) + MAT * 16 * 128 + p.rank * 2;
            const unsigned seq = (call % 1023u + 1u) << 6;
            GAS unsigned* next = (GAS unsigned*)(myws + NXT_OFF) + ((par * NR + dd) * 2 + MAT) * 32;
            auto publish = [&](int q, int e) {
                int j, t; piece_of(q, j, t);
                if (lane == 0)
                    __hip_atomic_store(dflag + (j >> 1) * 128 + t * 16 + (j & 1), (unsigned short)(seq | ((unsigned)e & 63u)), __ATOMIC_RELAXED, __HIP_MEMORY_SCOPE_SYSTEM);
                STAMP(1 + t);
            };
            static_assert(WIN >= 1 && WIN <= 4, "window");
            constexpr int F0 = WIN - 1;
            bool first = true, sent = true;
#pragma unroll 1
            while (q0 < NPM) {
#pragma unroll
                for (int h = 0; h < 2; ++h) {
                    if (h == 1 && q0 >= NPM) break;
                    int j, t; piece_of(q0, j, t);
                    asm volatile("" ::: "memory");
                    unsigned g; u64 sx;
                    asm volatile("s_mov_b64 %1, exec\n\ts_mov_b64 exec, 1\n\tglobal_atomic_add %0, %2, %3, off sc0 sc1\n\ts_mov_b64 exec, %1"
                                 : "=&v"(g), "=&s"(sx) : "v"(next), "v"(1u) : "memory");
                    load_piece(q2, b[h]);
                    asm volatile("" ::: "memory");
                    const bool early = EARLY && t == NT - 1;
                    const int off = (early ? olast : obase + t * 32 * 4096) + j * 4096 + lane * 16;
#pragma unroll
                    for (int f = 0; f < 4; ++f) {
                        if (f < WIN && sent) { if (f == 0) wait_vm<win_cnt(0) + 1>(); else if (f == 1) wait_vm<win_cnt(1) + 1>();
                                               else if (f == 2) wait_vm<win_cnt(2) + 1>(); else wait_vm<win_cnt(3) + 1>(); }
                        else if (f == 0) wait_vm<win_cnt(0)>(); else if (f == 1) wait_vm<win_cnt(1)>();
                        else if (f == 2) wait_vm<win_cnt(2)>(); else wait_vm<win_cnt(3)>();
                        if (f == F0 && !sent) publish(qp, e1);
                        __builtin_amdgcn_raw_buffer_store_b128(__builtin_bit_cast(u32x4, o[f]), rs, off + f * 1024, 0, ST_AUX);
                    }
                    sent = early;
                    if (early) publish(q0, e0);
                    if (first) __builtin_amdgcn_s_barrier();
                    if (first && lane == 0) {
                        __hip_atomic_store(ctr, call, __ATOMIC_RELAXED, __HIP_MEMORY_SCOPE_AGENT);
                        if (wi < 2) __hip_atomic_store((GAS unsigned*)(myws + NXT_OFF) + (((par ^ 1u) * NR + dd) * 2 + MAT) * 32, 0u, __ATOMIC_RELAXED, __HIP_MEMORY_SCOPE_SYSTEM);
                    }
                    first = false;
                    e1 = e0;
                    e0 = convert(b[h ^ 1]);
                    qp = q0; q0 = q1; q1 = q2;
                    asm volatile("s_waitcnt vmcnt(12)" : "+v"(g) :: "memory");
                    q2 = 3 * WPM + (int)min((unsigned)__builtin_amdgcn_readfirstlane(g), (unsigned)NPM);
                }
            }
            if (!sent) { wait_vm<0>(); publish(qp, e1); }
        };
        if (wi & 1) run(std::integral_constant<int, 1>{}); else run(std::integral_constant<int, 0>{});
        STAMP_DUMP();
        return;
    }

    if (wave * 64 + lane < 320) RQ[wave * 64 + lane] = 0;
    asm volatile("s_waitcnt lgkmcnt(0)" ::: "memory");
    __builtin_amdgcn_s_barrier();
    if (!(p.mode & 2)) { STAMP(0); STAMP_DUMP(); return; }

    const int tr = (xcd >> 1) * 4 + (sl >> 3), tc = (xcd & 1) * 8 + (sl & 7);

    if (wave >= NGEMM) {
        GAS unsigned* ctr = (GAS unsigned*)(myws + CTR_OFF) + wg * 16 + wave;
        const unsigned call = __hip_atomic_load(ctr, __ATOMIC_RELAXED, __HIP_MEMORY_SCOPE_AGENT) + 1u;
        if (lane == 0) __hip_atomic_store(ctr, call, __ATOMIC_RELAXED, __HIP_MEMORY_SCOPE_AGENT);
        const unsigned par = call & 1u;
        const char* data = myws + DATA_OFF + par * DATA_PAR;
        const char* dlast = myws + DATA_OFF + LAST_OFF + (call % 3u) * LAST_BUF;
        GAS unsigned* tmo = (GAS unsigned*)(myws + TMO_OFF);
        if (EARLY) {
            const __amdgpu_buffer_rsrc_t rz = __builtin_amdgcn_make_buffer_rsrc(myws + DATA_OFF + LAST_OFF + ((call + 2u) % 3u) * LAST_BUF, 0, (int)LAST_BUF, 0x00020000);
            const u32x4 z = {0u, 0u, 0u, 0u};
            const int zo = (wg * NLOAD + wave - NGEMM) * 2048 + lane * 16;
            __builtin_amdgcn_raw_buffer_store_b128(z, rz, zo, 0, 16);
            __builtin_amdgcn_raw_buffer_store_b128(z, rz, zo + 1024, 0, 16);
        }
        if (wave == NGEMM) {
            const __amdgpu_buffer_rsrc_t rf = __builtin_amdgcn_make_buffer_rsrc(myws + FLAG_OFF + par * FLAG_PAR, 0, (int)FLAG_PAR, 0x00020000);
            const bool up = lane >= 32;
            const int foff = (up ? 16 + tc : tr) * 256 + (lane & 31) * 8;
            const unsigned seq2 = (call % 1023u + 1u) * 0x00400040u;
            const u64 t0 = rtc();
            int spins = 0;
            u64 known = 0;
            auto poll = [&]() -> u32x2 {
                const u64 kk = known & (known >> 32);
                const bool want = !((kk >> (lane & 31)) & 1ull) && (!(lane & 16) || __builtin_popcountll(known) >= 12);
                asm volatile("" ::: "memory");
                return __builtin_amdgcn_raw_buffer_load_b64(rf, want ? foff : 0x40000000, 0, POLL_AUX);
            };
            auto digest = [&](const u32x2 fv) -> bool {
                const unsigned d0 = fv[0], d1 = fv[1];
                const unsigned rcv = (unsigned)__shfl((int)(up ? d0 : d1), lane ^ 32, 64);
                const unsigned xw = up ? rcv : d0, ww = up ? d1 : rcv;
                const u64 fresh = __ballot((xw & 0xFFC0FFC0u) == seq2 && (ww & 0xFFC0FFC0u) == seq2) & ~known;
                if (fresh) {
                    if ((fresh >> lane) & 1ull) {
                        const int x0 = (int)(xw << 26) >> 26, x1 = (int)(xw << 10) >> 26, w0 = (int)(ww << 26) >> 26, w1 = (int)(ww << 10) >> 26;
                        volatile LAS unsigned short* rq = (volatile LAS unsigned short*)RQ + lane * 2;
                        rq[0]   = (unsigned short)(18 - 8 * (x0 + w0)); rq[128] = (unsigned short)(18 - 8 * (x0 + w1));
                        rq[256] = (unsigned short)(18 - 8 * (x1 + w0)); rq[384] = (unsigned short)(18 - 8 * (x1 + w1));
                    }
                    known |= fresh;
                    asm volatile("" ::: "memory");
                    if (lane == 0) __hip_atomic_fetch_or(AVAIL, fresh, __ATOMIC_RELAXED, __HIP_MEMORY_SCOPE_WORKGROUP);
                }
                return known == ~0ull;
            };
            u32x2 fv[PD];
#pragma unroll
            for (int i = 0; i < PD - 1; ++i) fv[i] = poll();
            bool stop = false;
            while (!stop) {
#pragma unroll
                for (int i = 0; i < PD; ++i) {
                    fv[(i + PD - 1) % PD] = poll();
                    if (digest(fv[i])) { stop = true; break; }
                    if (__builtin_popcountll(known) < POLL_THR) __builtin_amdgcn_s_sleep(POLL_SLOW);
                    else __builtin_amdgcn_s_sleep(POLL_SLEEP);
                }
                if (!stop && (++spins & 31) == 0) {
                    if (__hip_atomic_load(tmo, __ATOMIC_RELAXED, __HIP_MEMORY_SCOPE_AGENT) == call || (long long)(rtc() - t0) > DEADLINE) {
                        if (lane == 0) { __hip_atomic_store(tmo, call, __ATOMIC_RELAXED, __HIP_MEMORY_SCOPE_AGENT); *ABORT = 1u; }
                        break;
                    }
                }
            }
            if (lane == 0) *FIN = 1u;
        }
        asm volatile("s_waitcnt vmcnt(0) lgkmcnt(0)" ::: "memory");
        int slot = -1;
        bool cur = false;
        unsigned curw = 0, cura = 0;
        auto issue = [&](int u, int sl_) {
            const int src = 2 * (u & 3) + (u >> 5), t = (u >> 2) & 7;
            const bool last = EARLY && t == NT - 1;
            const char* gx = (last ? dlast + (size_t)(src * 32) * 4096 : data + (size_t)((src * NT + t) * 32) * 4096) + 2 * tr * 4096 + lane * 16;
            const char* gw = gx + (last ? LAST_BUF / 2 : DATA_MAT) + (2 * tc - 2 * tr) * 4096;
#pragma unroll
            for (int q = 0; q < 8; ++q)
                __builtin_amdgcn_global_load_lds((const unsigned*)(gx + q * 1024), (unsigned*)(lds + sl_ * SLOT + q * 1024), 16, 0, LD_AUX);
#pragma unroll
            for (int q = 0; q < 8; ++q)
                __builtin_amdgcn_global_load_lds((const unsigned*)(gw + q * 1024), (unsigned*)(lds + sl_ * SLOT + 8192 + q * 1024), 16, 0, LD_AUX);
        };
        auto finish = [&]() {
            if (lane < 4) asm volatile("ds_write_b16 %0, %1 offset:2" :: "v"(cura), "v"(curw) : "memory");
        };
        for (;;) {
            u32x4 c;
            unsigned sw;
            asm volatile("ds_read_b128 %0, %2 offset:1088\n\tds_read_b32 %1, %3 offset:1024\n\ts_waitcnt lgkmcnt(0)"
                         : "=&v"(c), "=&v"(sw) : "v"(cbase), "v"(cbase + (unsigned)(lane & 15) * 4u) : "memory");
            const u64 avail = ((u64)(unsigned)__builtin_amdgcn_readfirstlane(c[1]) << 32) | (unsigned)__builtin_amdgcn_readfirstlane(c[0]);
            const unsigned fin = __builtin_amdgcn_readfirstlane(c[3]);
            if (slot < 0 && (avail || !cur)) {
                const unsigned fr = (unsigned)__ballot(sw == 0u) & ((1u << NSLOT) - 1u);
                if (fr) {
                    const int sc = __builtin_ctz(fr);
                    unsigned old; u64 sx;
                    asm volatile("s_mov_b64 %1, exec\n\ts_mov_b64 exec, 1\n\tds_cmpst_rtn_b32 %0, %2, %3, %4 offset:1024\n\ts_mov_b64 exec, %1\n\ts_waitcnt lgkmcnt(0)"
                                 : "=&v"(old), "=&s"(sx) : "v"(cbase + 4u * sc), "v"(0u), "v"((unsigned)NGEMM) : "memory");
                    if (__builtin_amdgcn_readfirstlane(old) == 0) slot = sc;
                }
            }
            if (avail && slot >= 0) {
                const unsigned alo = (unsigned)avail, ahi = (unsigned)(avail >> 32);
                const int ul = alo ? __builtin_ctz(alo) : 32, uh = ahi ? __builtin_ctz(ahi) : 32;
                const int u = ul <= uh ? ul : 32 + uh;
                u64 old, sx;
                asm volatile("s_mov_b64 %1, exec\n\ts_mov_b64 exec, 1\n\tds_and_rtn_b64 %0, %2, %3 offset:1088\n\ts_mov_b64 exec, %1\n\ts_waitcnt lgkmcnt(0)"
                             : "=&v"(old), "=&s"(sx) : "v"(cbase), "v"(~(1ull << u)) : "memory");
                const unsigned ow = __builtin_amdgcn_readfirstlane(u < 32 ? (unsigned)old : (unsigned)(old >> 32));
                if (!((ow >> (u & 31)) & 1u)) continue;
                issue(u, slot);
                if (cur) { wait_vm<16>(); finish(); }
                cur = true; curw = (unsigned)slot + 1u; cura = cbase + (unsigned)(lane * 64 + u) * 4u;
                slot = -1;
            } else if (cur) {
                wait_vm<0>(); finish(); cur = false;
            } else if ((fin >> 8) || (fin && !avail)) break;
            else __builtin_amdgcn_s_sleep(1);
        }
        STAMP_DUMP();
        return;
    }

    {
        STAMP(0);
        const int wr = wave >> 1, wc = wave & 1;
        f32x4 acc[4][4];
#pragma unroll
        for (int m = 0; m < 4; ++m)
#pragma unroll
            for (int c = 0; c < 4; ++c) acc[m][c] = (f32x4){0.f, 0.f, 0.f, 0.f};
        const __amdgpu_buffer_rsrc_t ro = __builtin_amdgcn_make_buffer_rsrc(p.out, 0, MM * NN * 4, 0x00020000);
        const int ooff = ((tr * 128 + wr * 64 + (lane >> 4) * 4) * NN + tc * 128 + wc * 64 + (lane & 15) * 4) * 4;
        bool flushed = false, aborted = false;
        GAS unsigned* ctr = (GAS unsigned*)(myws + CTR_OFF) + wg * 16 + wave;
        const unsigned call = __hip_atomic_load(ctr, __ATOMIC_RELAXED, __HIP_MEMORY_SCOPE_AGENT) + 1u;
        if (lane == 0) __hip_atomic_store(ctr, call, __ATOMIC_RELAXED, __HIP_MEMORY_SCOPE_AGENT);
        bool lost = false;
        int ucur = 0;
        const unsigned abase = (unsigned)(uintptr_t)L8 + (unsigned)(wr * 4096 + lane * 16) - SLOT;
        const unsigned bbase = (unsigned)(uintptr_t)L8 + (unsigned)(8192 + wc * 4096 + lane * 16) - SLOT;
        const unsigned qaddr = cbase + (unsigned)(wave * 64 + lane) * 4u;
        f16x8 fa[2][4], fb[2][4];
        u64 done = 0, pend = 0;
        unsigned q = 0;
#define LDS_FRAG(dst, addr, off) asm volatile("ds_read_b128 %0, %1 offset:" #off : "=v"(dst) : "v"(addr) : "memory")
#define FRAG_WAIT(a, b) asm volatile("s_waitcnt lgkmcnt(0)" : "+v"(a[0]), "+v"(a[1]), "+v"(a[2]), "+v"(a[3]), "+v"(b[0]), "+v"(b[1]), "+v"(b[2]), "+v"(b[3]) :: "memory")
#define MFMA(m, c, a, b) acc[m][c] = __builtin_amdgcn_mfma_f32_16x16x32_f16(a[m], b[c], acc[m][c], 0, 0, 0)
#define PIN() __builtin_amdgcn_sched_barrier(0)
        auto release = [&](unsigned s1, unsigned n) {
            u64 sx;
            asm volatile("s_mov_b64 %0, exec\n\ts_mov_b64 exec, 1\n\tds_sub_u32 %1, %2 offset:1020\n\ts_mov_b64 exec, %0"
                         : "=&s"(sx) : "v"(cbase + 4u * s1), "v"(n) : "memory");
        };
        auto take = [&](int u) {
            pend &= pend - 1; done |= 1ull << u; ucur = u;
        };
        auto fetch = [&](unsigned s1, f16x8* a, f16x8* b) {
            const unsigned va = abase + s1 * SLOT, vb = bbase + s1 * SLOT;
            LDS_FRAG(a[0], va, 0); LDS_FRAG(b[0], vb, 0); LDS_FRAG(b[1], vb, 1024); LDS_FRAG(b[2], vb, 2048); LDS_FRAG(b[3], vb, 3072);
            LDS_FRAG(a[1], va, 1024); LDS_FRAG(a[2], va, 2048); LDS_FRAG(a[3], va, 3072);
            release(s1, 1u);
            asm volatile("ds_read_b32 %0, %1" : "=v"(q) : "v"(qaddr) : "memory");
        };
        auto verify = [&](f16x8* a, f16x8* b) {
            u64 tv0 = 0;
            for (;;) {
                unsigned mn = ~0u;
#pragma unroll
                for (int i = 0; i < 4; ++i)
                    mn = min(mn, min(__builtin_bit_cast(u32x4, a[i])[0], __builtin_bit_cast(u32x4, b[i])[0]));
                if (__ballot(mn == 0u) == 0ull) return;
                if (!tv0) tv0 = rtc();
                else if ((long long)(rtc() - tv0) > DEADLINE) {
                    if (lane == 0) { __hip_atomic_store((GAS unsigned*)(myws + TMO_OFF), call, __ATOMIC_RELAXED, __HIP_MEMORY_SCOPE_AGENT); *ABORT = 1u; }
                    lost = true;
                    return;
                }
                const __amdgpu_buffer_rsrc_t rl = __builtin_amdgcn_make_buffer_rsrc(myws + DATA_OFF + LAST_OFF + (call % 3u) * LAST_BUF, 0, (int)LAST_BUF, 0x00020000);
                const int so = (2 * (ucur & 3) + (ucur >> 5)) * 32 * 4096 + lane * 16;
#pragma unroll
                for (int i = 0; i < 4; ++i) {
                    a[i] = __builtin_bit_cast(f16x8, __builtin_amdgcn_raw_buffer_load_b128(rl, so + (2 * tr + wr) * 4096 + i * 1024, 0, 17));
                    b[i] = __builtin_bit_cast(f16x8, __builtin_amdgcn_raw_buffer_load_b128(rl, so + (int)(LAST_BUF / 2) + (2 * tc + wc) * 4096 + i * 1024, 0, 17));
                }
            }
        };
        auto wait_unit = [&]() {
            while (!pend) {
                unsigned ab;
                asm volatile("ds_read_b32 %0, %2\n\tds_read_u8 %1, %3 offset:1101\n\ts_waitcnt lgkmcnt(0)"
                             : "=&v"(q), "=&v"(ab) : "v"(qaddr), "v"(cbase) : "memory");
                pend = __ballot((q >> 16) != 0u) & ~done;
                if (pend) break;
                if (__builtin_amdgcn_readfirstlane(ab) != 0) { aborted = true; break; }
                __builtin_amdgcn_s_sleep(1);
            }
        };
        while (done != ~0ull && !aborted) {
            wait_unit();
            if (aborted) break;
            int u = __builtin_ctzll(pend);
            unsigned w = (unsigned)__builtin_amdgcn_readlane((int)q, u);
            const unsigned ecw = w & 0xFFFFu;
            const int ecur = (int)(short)ecw;
            take(u);
            fetch(w >> 16, fa[0], fb[0]);
            bool more = true;
            while (more) {
#pragma unroll
                for (int h = 0; h < 2; ++h) {
                    if (more) {
                        f16x8* a = fa[h]; f16x8* b = fb[h]; f16x8* na = fa[h ^ 1]; f16x8* nb = fb[h ^ 1];
                        asm volatile("s_waitcnt lgkmcnt(0)" : "+v"(a[0]), "+v"(a[1]), "+v"(a[2]), "+v"(a[3]), "+v"(b[0]), "+v"(b[1]), "+v"(b[2]), "+v"(b[3]), "+v"(q) :: "memory");
                        if (EARLY && (ucur & 28) == 28) verify(a, b);
                        PIN();
                        MFMA(0, 0, a, b); MFMA(0, 1, a, b);
                        pend = __ballot((q >> 16) != 0u) & ~done;
                        bool got = false;
                        if (pend) {
                            u = __builtin_ctzll(pend);
                            w = (unsigned)__builtin_amdgcn_readlane((int)q, u);
                            got = (w & 0xFFFFu) == ecw;
                        }
                        if (got) take(u);
                        const unsigned s1 = got ? w >> 16 : 1u, va = abase + s1 * SLOT, vb = bbase + s1 * SLOT;
                        PIN();
                        MFMA(0, 2, a, b); LDS_FRAG(na[0], va, 0);    PIN();
                        MFMA(0, 3, a, b); LDS_FRAG(nb[0], vb, 0);    PIN();
                        MFMA(1, 0, a, b); LDS_FRAG(nb[1], vb, 1024); PIN();
                        MFMA(1, 1, a, b); LDS_FRAG(nb[2], vb, 2048); PIN();
                        MFMA(1, 2, a, b); LDS_FRAG(nb[3], vb, 3072); PIN();
                        MFMA(1, 3, a, b); LDS_FRAG(na[1], va, 1024); PIN();
                        MFMA(2, 0, a, b); LDS_FRAG(na[2], va, 2048); PIN();
                        MFMA(2, 1, a, b); LDS_FRAG(na[3], va, 3072); PIN();
                        MFMA(2, 2, a, b); release(s1, got ? 1u : 0u); PIN();
                        MFMA(2, 3, a, b); MFMA(3, 0, a, b); MFMA(3, 1, a, b); PIN();
                        if (got) asm volatile("ds_read_b32 %0, %1" : "=v"(q) : "v"(qaddr) : "memory");
                        PIN();
                        MFMA(3, 2, a, b); MFMA(3, 3, a, b);
                        PIN();
                        if (!got) {
                            if (pend || done == ~0ull) more = false;
                            else {
                                wait_unit();
                                if (aborted) more = false;
                                else {
                                    u = __builtin_ctzll(pend);
                                    w = (unsigned)__builtin_amdgcn_readlane((int)q, u);
                                    if ((w & 0xFFFFu) != ecw) more = false;
                                    else { take(u); fetch(w >> 16, na, nb); }
                                }
                            }
                        }
                    }
                }
            }
            asm volatile("s_waitcnt lgkmcnt(0)" : "+v"(q) :: "memory");
            if (done == ~0ull) { STAMP(1); }
            int oo = ooff;
            asm volatile("" : "+v"(oo));
#pragma unroll
            for (int m = 0; m < 4; ++m)
#pragma unroll
                for (int r = 0; r < 4; ++r) {
                    f32x4 v;
#pragma unroll
                    for (int c = 0; c < 4; ++c) { v[c] = __builtin_amdgcn_ldexpf(acc[m][c][r], -ecur); acc[m][c][r] = 0.f; }
                    if (flushed) v += __builtin_bit_cast(f32x4, __builtin_amdgcn_raw_buffer_load_b128(ro, oo + (m * 16 + r) * NN * 4, 0, OUT_AUX));
                    __builtin_amdgcn_raw_buffer_store_b128(__builtin_bit_cast(u32x4, v), ro, oo + (m * 16 + r) * NN * 4, 0, OUT_AUX);
                }
            flushed = true;
        }
        if (aborted || lost) {
            const f32x4 v = {__builtin_nanf(""), __builtin_nanf(""), __builtin_nanf(""), __builtin_nanf("")};
#pragma unroll
            for (int m = 0; m < 16; ++m)
                __builtin_amdgcn_raw_buffer_store_b128(__builtin_bit_cast(u32x4, v), ro, ooff + ((m >> 2) * 16 + (m & 3)) * NN * 4, 0, 0);
        }
    }
}

extern "C" void kernel_launch(int rank, int nranks, ncclComm_t comm,
                              hipStream_t stream,
                              void* const* d_in, const int* in_sizes, int n_in,
                              void* d_out, int out_size,
                              const int* in_dtypes, int out_dtype,
                              void* const* d_out_all,
                              void* const* d_ws_all, size_t ws_bytes) {
    (void)comm; (void)in_dtypes; (void)out_dtype; (void)d_out_all;
    if (nranks != NR || n_in != 2 || (unsigned)rank >= (unsigned)NR || ws_bytes < WS_NEED ||
        in_sizes[0] != MM * KSH || in_sizes[1] != KSH * NN || out_size != MM * NN) {
        fprintf(stderr, "kernel_launch: unexpected problem geometry\n");
        return;
    }
    HIP_CHECK(hipFuncSetAttribute((const void*)fused_gemm_ar, hipFuncAttributeMaxDynamicSharedMemorySize, LDS_BYTES));
    Params p;
    p.x = (const float*)d_in[0];
    p.w = (const float*)d_in[1];
    p.out = (float*)d_out;
    for (int i = 0; i < NR; ++i) p.ws[i] = (char*)d_ws_all[i];
    p.rank = rank;
    p.mode = 3;
    hipLaunchKernelGGL(fused_gemm_ar, dim3(NWG), dim3(NTHREADS), LDS_BYTES, stream, p);
    HIP_CHECK(hipGetLastError());
}
```
